```python
import numpy as np

kernel_name = "gpu_gemm_a2a_m4096_k4096_n4096_e4m3e4m3_dqsilu"


ABSMAX_RTOL = 0.03

COMPUTE_DTYPE = "float32"
EPILOGUE = "dequant_silu"
IN_QUANT = None
OUT_QUANT = None
IN_FP8 = "e4m3e4m3"
IN_DTYPES = {"x": "e4m3", "w_mat": "e4m3",
             "scale_x": "float32", "scale_w": "float32"}
WORLD_SIZE = 8

def _snap_fp8(x, mant_bits, max_abs, min_normal):
    a = np.abs(x.astype(np.float64))
    _m, _e = np.frexp(a)
    q = np.where(a >= min_normal, np.ldexp(1.0, _e - 1 - mant_bits),
                 min_normal * 2.0 ** -mant_bits)
    snapped = np.minimum(np.round(a / q) * q, max_abs)
    return (np.sign(x) * snapped).astype(np.float32)

def setup_inputs():

    x = _snap_fp8(np.random.randn(4096, 4096).astype(np.float32), 3, 448.0, 2.0 ** -6)
    w_mat = _snap_fp8(np.random.randn(4096, 4096).astype(np.float32), 3, 448.0, 2.0 ** -6)
    scale_x = (0.02 * (1.0 + np.random.rand(1))).astype(np.float32)
    scale_w = (0.02 * (1.0 + np.random.rand(1))).astype(np.float32)
    return {"x": x, "w_mat": w_mat, "scale_x": scale_x, "scale_w": scale_w}

def reference(x, w_mat, scale_x, scale_w):
    acc = (x @ w_mat).astype(np.float32)
    y = (acc * (scale_x * scale_w)).astype(np.float32)
    return (y / (1.0 + np.exp(-np.clip(y, -60.0, 60.0)))).astype(np.float32)
```

```cpp
#include <hip/hip_runtime.h>
#include <rccl/rccl.h>
#include <cstdio>

typedef int v2i __attribute__((ext_vector_type(2)));
typedef int v4i __attribute__((ext_vector_type(4)));
typedef int v8i __attribute__((ext_vector_type(8)));
typedef float v4f __attribute__((ext_vector_type(4)));
typedef unsigned v4u __attribute__((ext_vector_type(4)));
#define LAS __attribute__((address_space(3)))

constexpr int NR = 8;
constexpr int KD = 4096, ND = 4096;
constexpr int MS = 512, NS = 512;
constexpr int BM = 32, BN = 64;
constexpr int NCW = 4;
constexpr int STEPS = KD / NCW / 128;
constexpr int NTILE = 4;
constexpr int XB = BM * 128, WB = 128 * BN, SB = XB + WB;
constexpr int RING = 3;
constexpr int WAVE_LDS = RING * SB;
constexpr int SYNC_OFF = NCW * WAVE_LDS;
constexpr int LDS_BYTES = SYNC_OFF + 256;
constexpr int PROW = BN * 4 + 16;
constexpr unsigned long long DEADLINE_TICKS = 100000000ull;
constexpr int WS_CNT = 0;
constexpr int WS_TMO = 512;
constexpr int WS_FLAG = 1024;
#define STAMP(i) do {} while (0)

struct Args {
  const unsigned char* x; const unsigned char* w; const float* sx; const float* sw;
  float* out[NR]; unsigned* ws[NR];
  int rank; int nranks;
};

__device__ __forceinline__ unsigned rfl(unsigned v) { return __builtin_amdgcn_readfirstlane(v); }

#define DMA(V, SRD, OFF) "buffer_load_dwordx4 " V ", " SRD ", " OFF " offen lds\n\t"
#define M0STEP "s_add_u32 m0, m0, 0x400\n\t"
template <int STRIDE2>
__device__ __forceinline__ void issue4(unsigned lds, __amdgpu_buffer_rsrc_t srd, unsigned so, unsigned v0, unsigned v1) {
  unsigned t, keep;
  asm volatile("s_nop 4\n\ts_mov_b32 %1, m0\n\ts_mov_b32 m0, %2\n\ts_add_u32 %0, %4, %7\n\t"
               DMA("%5", "%3", "%4") M0STEP "s_nop 0\n\t"
               DMA("%6", "%3", "%4") M0STEP "s_nop 0\n\t"
               DMA("%5", "%3", "%0") M0STEP "s_nop 0\n\t"
               DMA("%6", "%3", "%0") "s_mov_b32 m0, %1"
               : "=&s"(t), "=&s"(keep) : "s"(lds), "s"(srd), "s"(so), "v"(v0), "v"(v1), "i"(STRIDE2) : "scc", "memory");
}

__device__ __forceinline__ unsigned lds_ld(unsigned addr) { return *(volatile LAS unsigned*)(size_t)addr; }

__device__ __forceinline__ bool lds_wait(unsigned addr, unsigned target) {
  if (rfl(lds_ld(addr)) >= target) return true;
  unsigned long long t0 = __builtin_amdgcn_s_memrealtime();
  while (rfl(lds_ld(addr)) < target) {
    if (__builtin_amdgcn_s_memrealtime() - t0 > DEADLINE_TICKS) return false;
    __builtin_amdgcn_s_sleep(1);
  }
  return true;
}

__global__ __launch_bounds__(512) void gemm_a2a(Args a) {
  __shared__ __attribute__((aligned(16))) unsigned char smem[LDS_BYTES];
  const unsigned ldsb = (unsigned)(size_t)(LAS unsigned char*)smem;
  const unsigned lane = threadIdx.x & 63;
  const unsigned wid = rfl(threadIdx.x >> 6);
  const unsigned grp = blockIdx.x & 7, j = blockIdx.x >> 3;
  const unsigned dst = (a.rank + grp) & 7;
  const unsigned syn = ldsb + SYNC_OFF;
  STAMP(0);
  auto tile_rb = [&](unsigned t) { return (j & 7) + 8 * ((t ^ (t >> 1)) & 1); };
  auto tile_cb = [&](unsigned t) { return (j >> 3) + 4 * (t >> 1); };

  if (wid < NCW) {
    __amdgpu_buffer_rsrc_t srx = __builtin_amdgcn_make_buffer_rsrc((void*)a.x, (short)0, MS * KD, 0x00020000);
    __amdgpu_buffer_rsrc_t srw = __builtin_amdgcn_make_buffer_rsrc((void*)a.w, (short)0, KD * ND, 0x00020000);
    const unsigned vx0 = (lane >> 3) * KD + (((lane & 7) ^ (lane >> 4)) << 4);
    const unsigned vx1 = (vx0 ^ 64) + 8 * KD;
    const unsigned vw0 = (lane >> 2) * ND + (((lane & 3) ^ ((lane >> 4) & 1)) << 4);
    const unsigned vw1 = (vw0 ^ 32) + 16 * ND;
    const unsigned wbase = ldsb + wid * WAVE_LDS;
    const unsigned k0 = wid * (KD / NCW);
    auto issue = [&](unsigned s) {
      unsigned t = s / STEPS, ks = s % STEPS, slot = s % RING;
      unsigned k = k0 + ks * 128;
      unsigned lds = wbase + slot * SB;
      unsigned xo = tile_rb(t) * BM * KD + k;
      unsigned wo = k * ND + dst * NS + tile_cb(t) * BN;
      issue4<16 * KD>(lds, srx, xo, vx0, vx1);
      issue4<32 * ND>(lds + XB, srw, wo, vw0, vw1);
      issue4<32 * ND>(lds + XB + 4096, srw, wo + 64 * ND, vw0, vw1);
    };
#pragma unroll 1
    for (unsigned s0 = 0; s0 < RING; ++s0) issue(s0);
    asm volatile("s_barrier" ::: "memory");
    const unsigned r = lane & 15, g = lane >> 4;
    const unsigned xa0 = r * 128 + ((g ^ (r >> 1)) << 4), xa1 = xa0 ^ 64;
    const unsigned q = r >> 1, p = lane & 1;
    const unsigned fw = ((q >> 2) & 1) | ((g & 1) << 1);
    const unsigned wa = XB + (16 * g + q) * 64 + 8 * p;
    unsigned issued = 3;
    unsigned s = 0;
#pragma unroll 1
    for (unsigned t = 0; t < NTILE; ++t) {
      v4f acc[2][4];
#pragma unroll
      for (int i = 0; i < 2; ++i)
#pragma unroll
        for (int n = 0; n < 4; ++n) acc[i][n] = v4f{0, 0, 0, 0};
#pragma unroll 1
      for (unsigned ks = 0; ks < STEPS; ++ks, ++s) {
        unsigned ahead = issued - s - 1;
        if (ahead >= 2) asm volatile("s_waitcnt vmcnt(24)" ::: "memory");
        else if (ahead == 1) asm volatile("s_waitcnt vmcnt(12)" ::: "memory");
        else asm volatile("s_waitcnt vmcnt(0)" ::: "memory");
        const unsigned sb = wbase + (s % RING) * SB;
        v8i xf[2], wf[4];
#pragma unroll
        for (int i = 0; i < 2; ++i) {
          v4i lo = *(const LAS v4i*)(size_t)(sb + xa0 + i * 2048);
          v4i hi = *(const LAS v4i*)(size_t)(sb + xa1 + i * 2048);
          xf[i] = v8i{lo[0], lo[1], lo[2], lo[3], hi[0], hi[1], hi[2], hi[3]};
        }
#pragma unroll
        for (int n = 0; n < 4; ++n) {
          unsigned ad = sb + wa + ((n ^ fw) << 4);
          v2i t0 = __builtin_amdgcn_ds_read_tr8_b64_v2i32((LAS v2i*)(size_t)(ad));
          v2i t1 = __builtin_amdgcn_ds_read_tr8_b64_v2i32((LAS v2i*)(size_t)(ad + 512));
          v2i t2 = __builtin_amdgcn_ds_read_tr8_b64_v2i32((LAS v2i*)(size_t)(ad + 4096));
          v2i t3 = __builtin_amdgcn_ds_read_tr8_b64_v2i32((LAS v2i*)(size_t)(ad + 4608));
          wf[n] = v8i{t0[0], t0[1], t1[0], t1[1], t2[0], t2[1], t3[0], t3[1]};
        }
        asm volatile("s_waitcnt lgkmcnt(0)" ::: "memory");
        if (ks == 0 && t > 0 && issued < NTILE * STEPS) {
          lds_wait(syn + 16 + 4 * (t - 1), 1);
          issue(issued); ++issued;
        }
        if (ks != STEPS - 1 && issued < NTILE * STEPS) { issue(issued); ++issued; }
#pragma unroll
        for (int n = 0; n < 4; ++n)
#pragma unroll
          for (int i = 0; i < 2; ++i)
            acc[i][n] = __builtin_amdgcn_mfma_scale_f32_16x16x128_f8f6f4(wf[n], xf[i], acc[i][n], 0, 0, 0, 0, 0, 0);
      }
      const unsigned pb = wbase + ((s - 1) % RING) * SB + r * PROW + g * 16;
#pragma unroll
      for (int i = 0; i < 2; ++i)
#pragma unroll
        for (int n = 0; n < 4; ++n) *(LAS v4f*)(size_t)(pb + i * 16 * PROW + n * 64) = acc[i][n];
      asm volatile("s_waitcnt lgkmcnt(0)" ::: "memory");
      if (lane == 0) __hip_atomic_fetch_add((LAS unsigned*)(size_t)(syn + 4 * t), 1u, __ATOMIC_RELAXED, __HIP_MEMORY_SCOPE_WORKGROUP);
      STAMP(1 + t);
    }
  } else {
    const unsigned t = wid - NCW;
    if (t == 0) { if (lane < 16) *(volatile LAS unsigned*)(size_t)(syn + 4 * lane) = 0; asm volatile("s_waitcnt lgkmcnt(0)" ::: "memory"); }
    asm volatile("s_barrier" ::: "memory");
    unsigned* myws = a.ws[a.rank];
    const unsigned call = rfl(__hip_atomic_load(&myws[WS_CNT + blockIdx.x], __ATOMIC_RELAXED, __HIP_MEMORY_SCOPE_AGENT)) + 1;
    const float scale = a.sx[0] * a.sw[0];
    float* op = a.out[dst] + (size_t)(a.rank * MS + tile_rb(t) * BM + (lane >> 4)) * NS + tile_cb(t) * BN + (lane & 15) * 4;
    bool ok = lds_wait(syn + 4 * t, NCW);
    STAMP(1);
    const unsigned pb = ldsb + ((t * STEPS + STEPS - 1) % RING) * SB + (lane >> 4) * PROW + (lane & 15) * 16;
    v4f v[8];
#pragma unroll
    for (int i = 0; i < 8; ++i) {
      v4f p0 = *(const LAS v4f*)(size_t)(pb + i * 4 * PROW);
      v4f p1 = *(const LAS v4f*)(size_t)(pb + i * 4 * PROW + WAVE_LDS);
      v4f p2 = *(const LAS v4f*)(size_t)(pb + i * 4 * PROW + 2 * WAVE_LDS);
      v4f p3 = *(const LAS v4f*)(size_t)(pb + i * 4 * PROW + 3 * WAVE_LDS);
      v[i] = (p0 + p1) + (p2 + p3);
    }
    asm volatile("s_waitcnt lgkmcnt(0)" ::: "memory");
    if (lane == 0) *(volatile LAS unsigned*)(size_t)(syn + 16 + 4 * t) = 1;
#pragma unroll
    for (int i = 0; i < 8; ++i) {
      v4f o;
#pragma unroll
      for (int e = 0; e < 4; ++e) {
        float y = v[i][e] * scale;
        float yc = fminf(fmaxf(y, -60.0f), 60.0f);
        o[e] = y * __builtin_amdgcn_rcpf(1.0f + __expf(-yc));
      }
      if (!ok) o = v4f{__builtin_nanf(""), __builtin_nanf(""), __builtin_nanf(""), __builtin_nanf("")};
      asm volatile("global_store_dwordx4 %0, %1, off sc0 sc1\n\ts_nop 1" :: "v"(op + (size_t)i * 4 * NS), "v"(o) : "memory");
    }
    STAMP(2);
    asm volatile("s_waitcnt vmcnt(0)" ::: "memory");
    STAMP(3);
    unsigned old = 0;
    if (lane == 0) old = __hip_atomic_fetch_add((LAS unsigned*)(size_t)(syn + 32), 1u, __ATOMIC_RELAXED, __HIP_MEMORY_SCOPE_WORKGROUP);
    old = rfl(old);
    if (old == NTILE - 1) {
      if (lane == 0) {
        if (grp != 0 && dst < (unsigned)a.nranks)
          __hip_atomic_store(&a.ws[dst][WS_FLAG + a.rank * 32 + j], call, __ATOMIC_RELAXED, __HIP_MEMORY_SCOPE_SYSTEM);
        __hip_atomic_store(&myws[WS_CNT + blockIdx.x], call, __ATOMIC_RELAXED, __HIP_MEMORY_SCOPE_AGENT);
      }
      if (blockIdx.x == 0) {
        const unsigned src = lane >> 3;
        const unsigned* f = myws + WS_FLAG + lane * 4;
        unsigned long long t0 = __builtin_amdgcn_s_memrealtime();
        bool pend = src != (unsigned)a.rank;
        bool timed_out = false;
        while (true) {
          if (pend) {
            v4u fv;
            asm volatile("global_load_dwordx4 %0, %1, off sc0 sc1\n\ts_waitcnt vmcnt(0)" : "=v"(fv) : "v"(f) : "memory");
            pend = (int)(fv[0] - call) < 0 || (int)(fv[1] - call) < 0 || (int)(fv[2] - call) < 0 || (int)(fv[3] - call) < 0;
          }
          if (__builtin_amdgcn_ballot_w64(pend) == 0) break;
          if (__builtin_amdgcn_s_memrealtime() - t0 > DEADLINE_TICKS) { timed_out = true; break; }
        }
        if (timed_out && lane == 0) {
          __hip_atomic_store(&myws[WS_TMO], call, __ATOMIC_RELAXED, __HIP_MEMORY_SCOPE_AGENT);
          a.out[a.rank][(size_t)a.rank * MS * NS] = __builtin_nanf("");
        }
        __builtin_amdgcn_fence(__ATOMIC_ACQUIRE, "");
        STAMP(4);
      }
    }
  }
}

extern "C" void kernel_launch(int rank, int nranks, ncclComm_t comm, hipStream_t stream, void* const* d_in, const int* in_sizes, int n_in,
                              void* d_out, int out_size, const int* in_dtypes, int out_dtype, void* const* d_out_all,
                              void* const* d_ws_all, size_t ws_bytes) {
  if (nranks != NR || rank < 0 || rank >= NR || n_in < 4 || in_sizes[0] != MS * KD || in_sizes[1] != KD * ND || out_size != NR * MS * NS) {
    fprintf(stderr, "unexpected problem shape\n");
    return;
  }
  Args a;
  a.x = (const unsigned char*)d_in[0]; a.w = (const unsigned char*)d_in[1];
  a.sx = (const float*)d_in[2]; a.sw = (const float*)d_in[3];
  for (int i = 0; i < NR; ++i) { a.out[i] = (float*)d_out_all[i]; a.ws[i] = (unsigned*)d_ws_all[i]; }
  a.rank = rank; a.nranks = nranks;
  gemm_a2a<<<256, 512, 0, stream>>>(a);
}
```

```cpp
#include <hip/hip_runtime.h>
#include <rccl/rccl.h>
#include <cstdio>

typedef int v2i __attribute__((ext_vector_type(2)));
typedef int v4i __attribute__((ext_vector_type(4)));
typedef int v8i __attribute__((ext_vector_type(8)));
typedef float v4f __attribute__((ext_vector_type(4)));
typedef float v16f __attribute__((ext_vector_type(16)));
typedef unsigned v4u __attribute__((ext_vector_type(4)));
#define LAS __attribute__((address_space(3)))

constexpr int NR = 8;
constexpr int KD = 4096, ND = 4096;
constexpr int MS = 512, NS = 512;
constexpr int BM = 32, BN = 128;
constexpr int NCW = 4;
constexpr int NSW = 4;
constexpr int HSTEPS = KD / NCW / 64;
constexpr int NTILE = 2;
constexpr int XSLOT = BM * 128, WSLOT = 64 * BN;
constexpr int WAVE_LDS = 2 * XSLOT + 3 * WSLOT;
constexpr int SYNC_OFF = NCW * WAVE_LDS;
constexpr int LDS_BYTES = SYNC_OFF + 256;
constexpr int PROW = BN * 4 + 16;
static_assert(BM * PROW <= WAVE_LDS, "partial tile must fit the wave's ring");
constexpr unsigned long long DEADLINE_TICKS = 100000000ull;
constexpr int WS_CNT = 0;
constexpr int WS_TMO = 512;
constexpr int WS_FLAG = 1024;
#define STAMP(i) do {} while (0)
#define ROT 1

struct Args {
  const unsigned char* x; const unsigned char* w; const float* sx; const float* sw;
  float* out[NR]; unsigned* ws[NR];
  int rank; int nranks;
};

__device__ __forceinline__ unsigned rfl(unsigned v) { return __builtin_amdgcn_readfirstlane(v); }

__device__ __forceinline__ void dma4(unsigned lds, __amdgpu_buffer_rsrc_t srd, unsigned so, const unsigned (&v)[4]) {
  unsigned keep;
  asm volatile("s_nop 4\n\ts_mov_b32 %0, m0\n\ts_mov_b32 m0, %1\n\ts_nop 0\n\t"
               "buffer_load_dwordx4 %4, %2, %3 offen lds\n\t"
               "buffer_load_dwordx4 %5, %2, %3 offen offset:1024 lds\n\t"
               "buffer_load_dwordx4 %6, %2, %3 offen offset:2048 lds\n\t"
               "buffer_load_dwordx4 %7, %2, %3 offen offset:3072 lds\n\t"
               "s_mov_b32 m0, %0"
               : "=&s"(keep) : "s"(lds), "s"(srd), "s"(so), "v"(v[0]), "v"(v[1]), "v"(v[2]), "v"(v[3]) : "memory");
}

__device__ __forceinline__ unsigned lds_ld(unsigned addr) { return *(volatile LAS unsigned*)(size_t)addr; }
__device__ __forceinline__ void lds_inc(unsigned addr) {
  __hip_atomic_fetch_add((LAS unsigned*)(size_t)addr, 1u, __ATOMIC_RELAXED, __HIP_MEMORY_SCOPE_WORKGROUP);
}

__device__ __forceinline__ bool lds_wait(unsigned addr, unsigned target) {
  if (rfl(lds_ld(addr)) >= target) return true;
  unsigned long long t0 = __builtin_amdgcn_s_memrealtime();
  while (rfl(lds_ld(addr)) < target) {
    if (__builtin_amdgcn_s_memrealtime() - t0 > DEADLINE_TICKS) return false;
    __builtin_amdgcn_s_sleep(1);
  }
  return true;
}

__global__ __launch_bounds__(512) void gemm_a2a(Args a) {
  __shared__ __attribute__((aligned(16))) unsigned char smem[LDS_BYTES];
  const unsigned ldsb = (unsigned)(size_t)(LAS unsigned char*)smem;
  const unsigned lane = threadIdx.x & 63;
  const unsigned wid = rfl(threadIdx.x >> 6);
  const unsigned grp = blockIdx.x & 7, j = blockIdx.x >> 3;
  const unsigned dst = (a.rank + grp) & 7;
  const unsigned syn = ldsb + SYNC_OFF;
  STAMP(0);
  const unsigned rb = j & 15;
  auto tile_cb = [&](unsigned t) { return (j >> 4) + 2 * t; };

  if (wid < NCW) {
    __amdgpu_buffer_rsrc_t srx = __builtin_amdgcn_make_buffer_rsrc((void*)a.x, (short)0, MS * KD, 0x00020000);
    __amdgpu_buffer_rsrc_t srw = __builtin_amdgcn_make_buffer_rsrc((void*)a.w, (short)0, KD * ND, 0x00020000);
    unsigned vx[4], vw[4];
#pragma unroll
    for (int i = 0; i < 4; ++i) {
      vx[i] = (8 * i + (lane >> 3)) * KD + (((lane & 7) ^ (4 * (i & 1) + (lane >> 4))) << 4) - 1024 * i;
      vw[i] = (8 * i + (lane >> 3)) * ND + (((lane & 7) ^ ((lane >> 4) << 1)) << 4) - 1024 * i;
    }
    const unsigned xs = ldsb + wid * WAVE_LDS, wsb = xs + 2 * XSLOT;
    const unsigned k0 = wid * (KD / NCW);
    const unsigned rot = ROT ? 2 * ((rb + 4 * (j >> 4) + grp) & 7) : 0;
    const unsigned m = lane & 31, h = lane >> 5;
    const unsigned xa = m * 128 + (((2 * h) ^ ((m >> 1) & 7)) << 4);
    const unsigned gg = lane >> 4, q = (lane & 15) >> 1;
    const unsigned wa = (32 * (gg >> 1) + q) * 128 + (gg & 1) * 16 + 8 * (lane & 1);
    const unsigned fq = (q >> 1) & 3;
#pragma unroll 1
    for (unsigned t = 0; t < NTILE; ++t) {
      const unsigned xo = rb * BM * KD + k0;
      const unsigned wo = k0 * ND + dst * NS + tile_cb(t) * BN;
      auto issue = [&](unsigned g) {
        unsigned kk = ((g + rot) & (HSTEPS - 1)) * 64;
        if (!(g & 1)) dma4(xs + ((g >> 1) & 1) * XSLOT, srx, xo + kk, vx);
        unsigned sl = wsb + (g % 3) * WSLOT;
        dma4(sl, srw, wo + kk * ND, vw);
        dma4(sl + 4096, srw, wo + (kk + 32) * ND, vw);
      };
      if (t > 0) lds_wait(syn + 16 + 4 * (t - 1), NSW);
#pragma unroll 1
      for (unsigned g = 0; g < 3; ++g) issue(g);
      if (t == 0) asm volatile("s_barrier" ::: "memory");
      v16f acc[4];
#pragma unroll
      for (int n = 0; n < 4; ++n)
#pragma unroll
        for (int e = 0; e < 16; ++e) acc[n][e] = 0.f;
#pragma unroll 1
      for (unsigned g = 0; g < HSTEPS; ++g) {
        if (g < HSTEPS - 2) asm volatile("s_waitcnt vmcnt(20)" ::: "memory");
        else if (g == HSTEPS - 2) asm volatile("s_waitcnt vmcnt(8)" ::: "memory");
        else asm volatile("s_waitcnt vmcnt(0)" ::: "memory");
        const unsigned xb = xs + ((g >> 1) & 1) * XSLOT + (xa ^ ((g & 1) << 6));
        const unsigned wb = wsb + (g % 3) * WSLOT + wa;
        v4i lo = *(const LAS v4i*)(size_t)(xb);
        v4i hi = *(const LAS v4i*)(size_t)(xb ^ 16);
        v8i xf = v8i{lo[0], lo[1], lo[2], lo[3], hi[0], hi[1], hi[2], hi[3]};
        v8i wf[4];
#pragma unroll
        for (int n = 0; n < 4; ++n) {
          unsigned ad = wb + ((n ^ fq) << 5);
          v2i t0 = __builtin_amdgcn_ds_read_tr8_b64_v2i32((LAS v2i*)(size_t)(ad));
          v2i t1 = __builtin_amdgcn_ds_read_tr8_b64_v2i32((LAS v2i*)(size_t)(ad + 1024));
          v2i t2 = __builtin_amdgcn_ds_read_tr8_b64_v2i32((LAS v2i*)(size_t)(ad + 2048));
          v2i t3 = __builtin_amdgcn_ds_read_tr8_b64_v2i32((LAS v2i*)(size_t)(ad + 3072));
          wf[n] = v8i{t0[0], t0[1], t1[0], t1[1], t2[0], t2[1], t3[0], t3[1]};
        }
        asm volatile("s_waitcnt lgkmcnt(0)" ::: "memory");
        acc[0] = __builtin_amdgcn_mfma_scale_f32_32x32x64_f8f6f4(wf[0], xf, acc[0], 0, 0, 0, 0, 0, 0);
        acc[1] = __builtin_amdgcn_mfma_scale_f32_32x32x64_f8f6f4(wf[1], xf, acc[1], 0, 0, 0, 0, 0, 0);
        __builtin_amdgcn_sched_barrier(0);
        if (g + 3 < HSTEPS) issue(g + 3);
        __builtin_amdgcn_sched_barrier(0);
        acc[2] = __builtin_amdgcn_mfma_scale_f32_32x32x64_f8f6f4(wf[2], xf, acc[2], 0, 0, 0, 0, 0, 0);
        acc[3] = __builtin_amdgcn_mfma_scale_f32_32x32x64_f8f6f4(wf[3], xf, acc[3], 0, 0, 0, 0, 0, 0);
      }
      const unsigned pb = xs + m * PROW + h * 16;
#pragma unroll
      for (int n = 0; n < 4; ++n)
#pragma unroll
        for (int i = 0; i < 4; ++i)
          *(LAS v4f*)(size_t)(pb + n * 128 + i * 32) = v4f{acc[n][4 * i], acc[n][4 * i + 1], acc[n][4 * i + 2], acc[n][4 * i + 3]};
      asm volatile("s_waitcnt lgkmcnt(0)" ::: "memory");
      if (lane == 0) lds_inc(syn + 4 * t);
      STAMP(1 + t);
    }
  } else {
    const unsigned u = wid - NCW;
    if (u == 0) { if (lane < 16) *(volatile LAS unsigned*)(size_t)(syn + 4 * lane) = 0; asm volatile("s_waitcnt lgkmcnt(0)" ::: "memory"); }
    asm volatile("s_barrier" ::: "memory");
    unsigned* myws = a.ws[a.rank];
    const unsigned call = rfl(__hip_atomic_load(&myws[WS_CNT + blockIdx.x], __ATOMIC_RELAXED, __HIP_MEMORY_SCOPE_AGENT)) + 1;
    const float scale = a.sx[0] * a.sw[0];
    const unsigned pb = ldsb + (8 * u + (lane >> 5)) * PROW + (lane & 31) * 16;
#pragma unroll 1
    for (unsigned t = 0; t < NTILE; ++t) {
      float* op = a.out[dst] + (size_t)(a.rank * MS + rb * BM + 8 * u + (lane >> 5)) * NS + tile_cb(t) * BN + (lane & 31) * 4;
      bool ok = lds_wait(syn + 4 * t, NCW);
      STAMP(1 + 2 * t);
      v4f v[4];
#pragma unroll
      for (int i = 0; i < 4; ++i) {
        v4f p0 = *(const LAS v4f*)(size_t)(pb + i * 2 * PROW);
        v4f p1 = *(const LAS v4f*)(size_t)(pb + i * 2 * PROW + WAVE_LDS);
        v4f p2 = *(const LAS v4f*)(size_t)(pb + i * 2 * PROW + 2 * WAVE_LDS);
        v4f p3 = *(const LAS v4f*)(size_t)(pb + i * 2 * PROW + 3 * WAVE_LDS);
        v[i] = (p0 + p1) + (p2 + p3);
      }
      asm volatile("s_waitcnt lgkmcnt(0)" ::: "memory");
      if (lane == 0) lds_inc(syn + 16 + 4 * t);
#pragma unroll
      for (int i = 0; i < 4; ++i) {
        v4f o;
#pragma unroll
        for (int e = 0; e < 4; ++e) {
          float y = v[i][e] * scale;
          float yc = fminf(fmaxf(y, -60.0f), 60.0f);
          o[e] = y * __builtin_amdgcn_rcpf(1.0f + __expf(-yc));
        }
        if (!ok) o = v4f{__builtin_nanf(""), __builtin_nanf(""), __builtin_nanf(""), __builtin_nanf("")};
        asm volatile("global_store_dwordx4 %0, %1, off sc0 sc1\n\ts_nop 1" :: "v"(op + (size_t)i * 2 * NS), "v"(o) : "memory");
      }
      STAMP(2 + 2 * t);
    }
    asm volatile("s_waitcnt vmcnt(0)" ::: "memory");
    STAMP(5);
    unsigned old = 0;
    if (lane == 0) old = __hip_atomic_fetch_add((LAS unsigned*)(size_t)(syn + 32), 1u, __ATOMIC_RELAXED, __HIP_MEMORY_SCOPE_WORKGROUP);
    old = rfl(old);
    if (old == NSW - 1) {
      if (lane == 0) {
        if (grp != 0 && dst < (unsigned)a.nranks)
          __hip_atomic_store(&a.ws[dst][WS_FLAG + a.rank * 32 + j], call, __ATOMIC_RELAXED, __HIP_MEMORY_SCOPE_SYSTEM);
        __hip_atomic_store(&myws[WS_CNT + blockIdx.x], call, __ATOMIC_RELAXED, __HIP_MEMORY_SCOPE_AGENT);
      }
      if (blockIdx.x == 0) {
        const unsigned src = lane >> 3;
        const unsigned* f = myws + WS_FLAG + lane * 4;
        unsigned long long t0 = __builtin_amdgcn_s_memrealtime();
        bool pend = src != (unsigned)a.rank;
        bool timed_out = false;
        while (true) {
          if (pend) {
            v4u fv;
            asm volatile("global_load_dwordx4 %0, %1, off sc0 sc1\n\ts_waitcnt vmcnt(0)" : "=v"(fv) : "v"(f) : "memory");
            pend = (int)(fv[0] - call) < 0 || (int)(fv[1] - call) < 0 || (int)(fv[2] - call) < 0 || (int)(fv[3] - call) < 0;
          }
          if (__builtin_amdgcn_ballot_w64(pend) == 0) break;
          if (__builtin_amdgcn_s_memrealtime() - t0 > DEADLINE_TICKS) { timed_out = true; break; }
        }
        if (timed_out && lane == 0) {
          __hip_atomic_store(&myws[WS_TMO], call, __ATOMIC_RELAXED, __HIP_MEMORY_SCOPE_AGENT);
          a.out[a.rank][(size_t)a.rank * MS * NS] = __builtin_nanf("");
        }
        __builtin_amdgcn_fence(__ATOMIC_ACQUIRE, "");
        STAMP(6);
      }
    }
  }
}

extern "C" void kernel_launch(int rank, int nranks, ncclComm_t comm, hipStream_t stream, void* const* d_in, const int* in_sizes, int n_in,
                              void* d_out, int out_size, const int* in_dtypes, int out_dtype, void* const* d_out_all,
                              void* const* d_ws_all, size_t ws_bytes) {
  if (nranks != NR || rank < 0 || rank >= NR || n_in < 4 || in_sizes[0] != MS * KD || in_sizes[1] != KD * ND || out_size != NR * MS * NS) {
    fprintf(stderr, "unexpected problem shape\n");
    return;
  }
  Args a;
  a.x = (const unsigned char*)d_in[0]; a.w = (const unsigned char*)d_in[1];
  a.sx = (const float*)d_in[2]; a.sw = (const float*)d_in[3];
  for (int i = 0; i < NR; ++i) { a.out[i] = (float*)d_out_all[i]; a.ws[i] = (unsigned*)d_ws_all[i]; }
  a.rank = rank; a.nranks = nranks;
  gemm_a2a<<<256, 512, 0, stream>>>(a);
}
```

```cpp
#include <hip/hip_runtime.h>
#include <rccl/rccl.h>
#include <cstdio>

typedef int v2i __attribute__((ext_vector_type(2)));
typedef int v4i __attribute__((ext_vector_type(4)));
typedef int v8i __attribute__((ext_vector_type(8)));
typedef float v4f __attribute__((ext_vector_type(4)));
typedef float v16f __attribute__((ext_vector_type(16)));
typedef unsigned v2u __attribute__((ext_vector_type(2)));
typedef unsigned v4u __attribute__((ext_vector_type(4)));
typedef unsigned v8u __attribute__((ext_vector_type(8)));
typedef unsigned v16u __attribute__((ext_vector_type(16)));
#define LAS __attribute__((address_space(3)))
#define GAS __attribute__((address_space(1)))

constexpr int NR = 8;
constexpr int KD = 4096, ND = 4096;
constexpr int MS = 512, NS = 512;
constexpr int BN = 128;
constexpr int NW = 4;
#define FROWS 96
constexpr int RROWS = MS - FROWS;
constexpr int NRW = 28;
constexpr int NFW = 4;
constexpr int NCH = FROWS * 4 / (NFW * NW);
constexpr int MPF = 20;
static_assert(FROWS % 8 == 0 && RROWS <= 448 && RROWS > 384, "bad split");
constexpr int PIECE = 4096;
constexpr int WAVE_LDS = 10 * PIECE;
constexpr int LDS_BYTES = NW * WAVE_LDS;
constexpr int PROW = BN * 4 + 16;
static_assert(64 * PROW <= WAVE_LDS, "partial tile must fit the wave's ring");
constexpr unsigned long long DEADLINE_TICKS = 100000000ull;
constexpr int WS_CNT = 0;
constexpr int WS_TMO = 512;
constexpr int WS_FLAG = 1024;
constexpr int WS_FWD = 2048;
constexpr size_t WS_STAGE = 2u << 20;
constexpr int STAGE_BYTES = NR * FROWS * KD;
constexpr int WAITER = 31 * 8;
#define STAMP(i) do {} while (0)
#define LSTAMP(i) do {} while (0)
#define TLBW 1
#define PT0 0
#define PTW 10
#define PTR 0
#define PTN 13
#define DO 0
#define QX 1
#define XF 0
#define WRM 9
#define WX 1
#define WW 16
#define WFL 8
#define XSW 5
#define SLOT0 720
#define SLOTK 108
#define SLN 14
#define SLD 60
#define SLS 5
#define TL "aaaaaaaAAAbBBBBBBcCCCCCDCDDdDDD"
#define AS "ikkkllmlllnmlmmnnnjjljlhkkjjkllllllmmmmmmmmmmmmm"
#define AT "ljljnknmpmmlopkliholllkjhjjiieehmmmmmmmmmmmmmmmm"
#define WN0 0
#define WN1 30
#define UP 1
#define NA 10
#define RD "44245427548754cgedbe"
#define LN 14
#define LEADH 960
#define LEADS 12
#define SH1 25
#define SHS 4
#define SH2 180
#define SHT 12
#define DQ64 8
#define XO 0
#define WF 152
#define WIN 5
#define Q0 0
#define Q1 0
#define PF3 0
#define XROT 1
#define FWW 6
#define BADR 1
#define CPF 1
#define CPL 88
#define SKT 1
#define EW2 0
#define DQS 8
#define DQQ 9
#define PFW 0
#define PW ""
#define INV 0
#define PD ""
#define PF ""
#define PS " sc0 sc1"
#define PFS " sc0 sc1"
#define LEAD 825
#define LST 1100
#define NAR 0
#define FSW 0
#define ZR 0
#define DLY0 1000

struct Args {
  const unsigned char* x; const unsigned char* w; const float* sx; const float* sw;
  int rank; int nranks; unsigned cfg, cfg2;
  struct { unsigned* ws; float* out; } peer[NR];
  unsigned desc[256];
  unsigned slot[32][2];
  unsigned short pf[MPF * NW][16];
};
static_assert(offsetof(Args, peer) == 0x30 && offsetof(Args, desc) == 0xb0 && offsetof(Args, slot) == 0x4b0 && offsetof(Args, pf) == 0x5b0 && sizeof(Args) <= 4096, "argument layout");
constexpr int fwd_wait(int i) {
  int n = 0;
  if (i < FWW) n = (FWW < NCH ? FWW : NCH) - 1 - i;
  for (int k = i < FWW ? 0 : i - FWW + 1; k < i; ++k) n += 1 + (k + FWW < NCH);
  return n;
}

__device__ __forceinline__ unsigned rfl(unsigned v) { return __builtin_amdgcn_readfirstlane(v); }

#define DMA2(POL) \
  asm volatile("s_mov_b32 m0, %0\n\ts_nop 0\n\t" \
               "buffer_load_dwordx4 %3, %1, %2 offen offset:%5 " POL " lds\n\t" \
               "buffer_load_dwordx4 %4, %1, %2 offen offset:%6 " POL " lds" \
               :: "s"(lds), "s"(srd), "s"(so), "v"(v[2 * H]), "v"(v[2 * H + 1]), "n"(2048 * H), "n"(2048 * H + 1024) : "memory", "m0")
template <int H> __device__ __forceinline__ void dma2(unsigned lds, __amdgpu_buffer_rsrc_t srd, unsigned so, const unsigned (&v)[4]) { DMA2(""); }
template <int H> __device__ __forceinline__ void dma2_w(unsigned lds, __amdgpu_buffer_rsrc_t srd, unsigned so, const unsigned (&v)[4]) { DMA2(PD); }
template <int H> __device__ __forceinline__ void dma2_sys(unsigned lds, __amdgpu_buffer_rsrc_t srd, unsigned so, const unsigned (&v)[4]) { DMA2("sc0 sc1"); }
template <int N> __device__ __forceinline__ void vmwait() { asm volatile("s_waitcnt vmcnt(%0)" :: "n"(N) : "memory"); }

__global__ __launch_bounds__(256) void gemm_a2a(Args a) {
  __shared__ __attribute__((aligned(16))) unsigned char smem[LDS_BYTES];
  const unsigned ldsb = (unsigned)(size_t)(LAS unsigned char*)smem;
  const unsigned lane = threadIdx.x & 63;
  const unsigned wid = rfl(threadIdx.x >> 6);
  const unsigned long long t_start = __builtin_amdgcn_s_memrealtime();
  const unsigned bid = blockIdx.x ^ XSW, grp = bid & 7, j = bid >> 3;
  if (INV == 1 && j == 0 && wid == 0) asm volatile("buffer_inv sc1" ::: "memory");
  if (INV == 2 && j == 0 && wid == 0) asm volatile("buffer_inv sc0 sc1" ::: "memory");
  v16u ka;
  v4u kp;
  v8u pf;
  unsigned desc;
  v2u sl2;
  asm volatile("s_load_dwordx16 %0, %5, 0x0\n\ts_load_dwordx4 %1, %5, %6 offset:0x30\n\ts_load_dword %2, %5, %7 offset:0xb0\n\ts_load_dwordx8 %3, %5, %8\n\ts_load_dwordx2 %4, %5, %9 offset:0x4b0"
               : "=&s"(ka), "=&s"(kp), "=&s"(desc), "=&s"(pf), "=&s"(sl2)
               : "s"(__builtin_amdgcn_kernarg_segment_ptr()), "s"(grp * 16), "s"(bid * 4),
                 "s"(j >= 32 - MPF ? 0x5b0 + ((31 - j) * NW + wid) * 32 : 0), "s"(j * 8) : "memory");
  float cpf = 0.f;
  if (CPF && j == 31 - CPF && threadIdx.x < CPL) {
    unsigned long long pc;
    asm volatile("s_getpc_b64 %0" : "=s"(pc));
    asm volatile("global_load_dword %0, %1, %2" : "=v"(cpf) : "v"(threadIdx.x * 128u), "s"(pc & ~255ull) : "memory");
  }
  unsigned vx[4], vw[4], vn[4];
#pragma unroll
  for (int i = 0; i < 4; ++i) {
    vx[i] = (8 * i + (lane >> 3)) * KD + (((lane & 7) ^ (4 * (i & 1) + (lane >> 4))) << 4) - 1024 * i;
    vw[i] = (8 * i + (lane >> 3)) * ND + (((lane & 7) ^ ((lane >> 4) << 1)) << 4) - 1024 * i;
    vn[i] = (16 * i + (lane >> 2)) * ND + (((lane & 3) ^ ((lane >> 3) & 2)) << 4) - 1024 * i;
  }
  const unsigned wsb = ldsb + wid * WAVE_LDS, xs = wsb + 6 * PIECE;
  const unsigned m = lane & 31, h = lane >> 5, q = (lane & 15) >> 1;
  unsigned xa = xs + m * 128 + (((2 * h) ^ ((m >> 1) & 7)) << 4), pw = wsb + m * PROW + h * 16, wa[4];
#pragma unroll
  for (int n = 0; n < 4; ++n) wa[n] = wsb + h * PIECE + q * 128 + ((lane >> 4) & 1) * 16 + 8 * (lane & 1) + ((n ^ ((q >> 1) & 3)) << 5);
  unsigned na = wsb + h * 2048 + q * 64 + ((lane >> 4) & 1) * 16 + 8 * (lane & 1) + ((q >> 2) << 5);
  const unsigned tw = threadIdx.x & 192;
  unsigned vo0 = j < NFW ? RROWS * KD + j * (NW * NCH * 1024) + tw * (NCH * 16) + (WRM & 8 ? lane * 16 + 4096 : (lane & (WFL - 1)) * 128)
                          : (tw * 16 + (((2 * XROT * grp) * 64 + (j - NFW) * WW + (lane & (WW - 1))) & 1023)) * ND;
  unsigned vo1 = j < NFW ? (WS_CNT + bid) * 4 : m * KD + tw * 16 + ((XROT * grp + h * WX) & 7) * 128;
  asm volatile("" : "+v"(xa), "+v"(pw), "+v"(na), "+v"(wa[0]), "+v"(wa[1]), "+v"(wa[2]), "+v"(wa[3]), "+v"(vo0), "+v"(vo1));
  asm volatile("s_waitcnt lgkmcnt(0)" : "+s"(ka), "+s"(kp), "+s"(desc), "+s"(pf), "+s"(sl2), "+v"(vx[0]), "+v"(vx[1]), "+v"(vx[2]), "+v"(vx[3]), "+v"(vw[0]), "+v"(vw[1]), "+v"(vw[2]), "+v"(vw[3]), "+v"(vn[0]), "+v"(vn[1]), "+v"(vn[2]), "+v"(vn[3]) :: "memory");
  unsigned slots = sl2[0], starts = sl2[1];
  auto ptr = [](unsigned lo, unsigned hi) { return (char*)(GAS char*)(((unsigned long long)hi << 32) | lo); };
  const unsigned char *xg = (const unsigned char*)ptr(ka[0], ka[1]), *wg = (const unsigned char*)ptr(ka[2], ka[3]);
  const unsigned rank = ka[8], nranks = ka[9];
  unsigned *myws = (unsigned*)ptr(ka[12], ka[13]), *peerws = (unsigned*)ptr(kp[0], kp[1]);
  float *myout = (float*)ptr(ka[14], ka[15]), *peerout = (float*)ptr(kp[2], kp[3]);
  const unsigned cx = ((ka[2] >> 9) + rank + grp) & 1 ? 0 : 3 ^ ((ka[10] >> 8) & 3);
  unsigned cv = 0;
  v4i fx[NCH];
  if ((WRM & 8) && grp != 0 && j < NFW) {
    asm volatile("global_load_dword %0, %1, %2 sc1" : "+v"(cv) : "v"(vo1), "s"(myws) : "memory");
#pragma unroll
    for (int i = 0; i < FWW; ++i) asm volatile("global_load_dwordx4 %0, %1, %2 offset:%3" PF : "=v"(fx[i]) : "v"(vo0), "s"(xg), "n"(1024 * i - 4096) : "memory");
  } else
  if (WRM && grp != 0 && j < (ka[10] & 255)) {
    const unsigned char* b0 = j < NFW ? xg : wg + (((rank + grp) & 7) * NS + ((((desc >> 8) & 3) ^ cx) * BN));
    const unsigned char* b1 = j < NFW ? (const unsigned char*)myws : xg + ((desc >> 1) & 31) * (32 * KD);
    if (WRM & 4) asm volatile("global_load_dword %0, %1, %2" : "+v"(cpf) : "v"(vo0), "s"(b0) : "memory");
    asm volatile("global_load_dword %0, %1, %2 sc1" : "+v"(cv) : "v"(vo1), "s"(b1) : "memory");
    if (WRM & 2) asm volatile("global_load_dword %0, %1, %2" : "+v"(cpf) : "v"(vo0), "s"(b0) : "memory");
  }
  __builtin_amdgcn_sched_barrier(0);
  unsigned pl = ka[12] + (WS_CNT + bid) * 4, ph = ka[13];
  asm("v_writelane_b32 %0, %2, 0\n\tv_writelane_b32 %1, %3, 0\n\tv_writelane_b32 %0, %4, 1\n\tv_writelane_b32 %1, %5, 1"
      : "+v"(pl), "+v"(ph) : "s"(ka[4]), "s"(ka[5]), "s"(ka[6]), "s"(ka[7]));
  const char* psc = ptr(pl, ph);
#define EVT(t) do {} while (0)
#define EVS() do {} while (0)
  const bool peer_ok = ((rank + grp) & 7) < nranks;
  unsigned call = 0;

  if (__builtin_expect(grp != 0 && j < NFW, 0)) {
    const unsigned off = (j * NW + wid) * (NCH * 1024) + lane * 16;
    const unsigned char* xp = xg + RROWS * KD + off;
#define FLD(i) asm volatile("global_load_dwordx4 %0, %1, off offset:%2" PF : "=v"(fx[i]) : "v"(xp + 4096 * ((i) >> 2)), "n"(1024 * ((i) & 3)) : "memory")
    if (!WRM) asm volatile("global_load_dword %0, %1, off sc1" : "=v"(cv) : "v"(myws + WS_CNT + bid) : "memory");
#pragma unroll
    for (int i = 0; i < FWW; ++i) if (!(WRM & 8)) FLD(i);
    STAMP(12);
    asm volatile("s_waitcnt vmcnt(%1)" : "+v"(cv) : "n"(FWW) : "memory");
    STAMP(13);
    call = rfl(cv) + 1;
    char* sp = (char*)peerws + WS_STAGE + (size_t)(call & 1) * STAGE_BYTES + (7 - grp) * (FROWS * KD) + off;
    if (!peer_ok) sp = (char*)myws + WS_STAGE + 2 * (size_t)STAGE_BYTES + lane * 16;
#pragma unroll
    for (int i = 0; i < NCH; ++i) {
      asm volatile("s_waitcnt vmcnt(%3)\n\tglobal_store_dwordx4 %0, %1, off offset:%2" PFS "\n\ts_nop 1"
                   :: "v"(sp + 4096 * (i >> 2)), "v"(fx[i]), "n"(1024 * (i & 3)), "n"(fwd_wait(i)) : "memory");
      if (i + FWW < NCH) FLD(i + FWW);
    }
    asm volatile("s_waitcnt vmcnt(0)" ::: "memory");
    if (lane == 0 && peer_ok) __hip_atomic_store(&peerws[WS_FWD + (8 - grp) * 64 + j * NW + wid], call, __ATOMIC_RELAXED, __HIP_MEMORY_SCOPE_SYSTEM);
    STAMP(7);
    if (ka[10] & 1024) {
      float d[4];
      const unsigned char* w3 = wg + (size_t)((j * NW + wid) * 256 + lane) * ND + ((rank + grp) & 7) * NS + (3 ^ cx) * BN;
#pragma unroll
      for (int c = 0; c < 4; ++c) asm volatile("global_load_dword %0, %1, off" PW : "=v"(d[c]) : "v"(w3 + c * 64 * ND) : "memory");
      asm volatile("s_waitcnt vmcnt(0)" :: "v"(d[0]), "v"(d[1]), "v"(d[2]), "v"(d[3]) : "memory");
      STAMP(5);
    }
  }
  if (__builtin_expect(grp == 0, 0)) {
    call = rfl(__hip_atomic_load(&myws[WS_CNT + bid], __ATOMIC_RELAXED, __HIP_MEMORY_SCOPE_AGENT)) + 1;
    while (__builtin_amdgcn_s_memrealtime() - t_start < (ka[10] >> 16)) __builtin_amdgcn_s_sleep(8);
    const unsigned char* wp = wg + (size_t)((j * NW + wid) * 32 + (lane >> 2)) * ND + rank * NS + (lane & 3) * 128;
    float d0, d1;
    asm volatile("global_load_dword %0, %2, off" PW "\n\tglobal_load_dword %1, %3, off" PW "\n\ts_waitcnt vmcnt(0)" : "=&v"(d0), "=&v"(d1) : "v"(wp), "v"(wp + 16 * ND) : "memory");
  }
  if (__builtin_expect(grp != 0 && j >= (ka[10] & 255), 0)) {
    const unsigned vxo = (lane >> 2) * KD + (lane & 3) * (KD / NW), vwo = lane * ND;
    float& d = cpf;
    if (j == 31 && wid == 2) asm volatile("global_load_dword %0, %1, off" : "+v"(d) : "v"(psc) : "memory");
    unsigned gate = (ka[11] & 511) + wid * ((ka[11] >> 9) & 127);
#pragma unroll 1
    for (unsigned n = 0; n < 16; ++n) {
      const unsigned e = pf[n >> 1] >> (16 * (n & 1)), kind = e & 7, p = ((e >> 3) + XROT * grp) & 7, r = (e >> 6) & 31;
      if (kind == 0) break;
      if (n < ka[11] >> 26) while ((unsigned)(__builtin_amdgcn_s_memrealtime() - t_start) < gate) __builtin_amdgcn_s_sleep(1);
      gate += (ka[11] >> 16) & 1023;
      const unsigned col = kind == 6 ? rank * NS + ((grp - 1) & 3) * BN : ((rank + grp) & 7) * NS + (((kind - 2) ^ cx) & 3) * BN;
      const unsigned char* pa = kind == 1 ? xg + (r * (16 * KD) + p * 128) : wg + ((((r >> 1) & 3) * (KD / NW) + p * 128 + (r & 1) * 64) * ND + col);
      asm volatile("s_bitcmp1_b32 %3, 11\n\ts_cbranch_scc1 1f\n\tglobal_load_dword %0, %1, %2\n\ts_branch 2f\n1:\n\tglobal_load_dword %0, %1, %2 nt\n2:"
                   : "+v"(d) : "v"(kind == 1 ? vxo : vwo), "s"(pa), "s"(e) : "memory", "scc");
      if (TLBW == 1 && n == 4 && j == 31 && wid == 3 && peer_ok) {
        asm volatile("global_load_dword %0, %1, off sc0 sc1" : "+v"(d)
                     : "v"(peerout + (size_t)rank * MS * NS + lane * (MS * NS / 64)) : "memory");
      }
    }
    if (TLBW == 2 && j == 31 && wid == 3 && peer_ok)
      asm volatile("global_load_dword %0, %1, off sc0 sc1" : "+v"(d) : "v"(peerout + (size_t)rank * MS * NS + lane * (MS * NS / 64)) : "memory");
    if (PFW) asm volatile("s_waitcnt vmcnt(0)" : "+v"(d) :: "memory");
    STAMP(7);
  }

  __amdgpu_buffer_rsrc_t srw = __builtin_amdgcn_make_buffer_rsrc((void*)wg, (short)0, KD * ND, 0x00020000);
  const unsigned k0 = wid * (KD / NW);
  float* op0 = nullptr;
  bool flag = grp != 0 && peer_ok;
#pragma unroll 1
  while (desc) {
    const unsigned nar = desc & (desc >> 7) & NAR, staged = (desc & 1) ^ nar, row0 = ((desc >> 1) & 31) * 32, nrows = ((desc >> 10) & 7) * 8 + 8, cb = (desc >> 8) & 3, rot = (desc >> 13) & 7;
    const unsigned dg = desc & 128 ? grp : 0;
    const bool full = nrows > 32 || staged;
    bool bad = false;
    const unsigned char* xb = xg;
    {
      const unsigned st = grp && !staged ? starts & 0xffff : 0;
      if (!SKT || st) while ((unsigned)(__builtin_amdgcn_s_memrealtime() - t_start) < st) __builtin_amdgcn_s_sleep(4);
    }
    if (staged) {
      const unsigned so = (row0 + ((lane >> 2) & 1) * (nrows - 1)) / FROWS + 1;
      const unsigned* f = myws + WS_FWD + so * 64 + (lane & 3) * 4;
      bool pend = lane < 8 && ((rank + so) & 7) < nranks;
      unsigned long long t0 = __builtin_amdgcn_s_memrealtime();
      while (true) {
        if (pend) {
          v4u fv;
          asm volatile("global_load_dwordx4 %0, %1, off sc0 sc1\n\ts_waitcnt vmcnt(0)" : "=v"(fv) : "v"(f) : "memory");
          pend = (int)(fv[0] - call) < 0 || (int)(fv[1] - call) < 0 || (int)(fv[2] - call) < 0 || (int)(fv[3] - call) < 0;
        }
        if (__builtin_amdgcn_ballot_w64(pend) == 0) break;
        if (__builtin_amdgcn_s_memrealtime() - t0 > DEADLINE_TICKS) { bad = true; break; }
        __builtin_amdgcn_s_sleep(8);
      }
      if (bad && lane == 0) __hip_atomic_store(&myws[WS_TMO], call, __ATOMIC_RELAXED, __HIP_MEMORY_SCOPE_AGENT);
      if (flag) {
        asm volatile("s_waitcnt vmcnt(0)\n\ts_barrier" ::: "memory");
        if (threadIdx.x == 0) __hip_atomic_store(&peerws[WS_FLAG + (8 - grp) * 32 + j], call, __ATOMIC_RELAXED, __HIP_MEMORY_SCOPE_SYSTEM);
        flag = false; LSTAMP(1);
      }
      xb = (const unsigned char*)myws + WS_STAGE + (size_t)(call & 1) * STAGE_BYTES;
    }
    if (staged) EVT(__builtin_amdgcn_s_memrealtime());
    STAMP(8);
    xb += row0 * KD;
    __amdgpu_buffer_rsrc_t sx0 = __builtin_amdgcn_make_buffer_rsrc((void*)xb, (short)0, nrows * KD, 0x00020000);
    __amdgpu_buffer_rsrc_t sx1 = __builtin_amdgcn_make_buffer_rsrc((void*)(xb + 32 * KD), (short)0, nrows > 32 ? (nrows - 32) * KD : 0, 0x00020000);
    const unsigned co = (dg ? cb ^ cx : cb) * BN + (desc & 64);
    const unsigned wo = k0 * ND + ((rank + dg) & 7) * NS + co;
    auto woff = [&](unsigned s) { return wo + ((s + 2 * rot) & 15) * (64 * ND); };
    auto xoff = [&](unsigned p) { return k0 + ((p + rot) & 7) * 128; };
#define DW(I, o, slot) dma2_w<(I) & 1>(wsb + (slot) * (2 * PIECE) + ((I) >> 1) * PIECE, srw, (o) + ((I) >> 1) * (32 * ND), vw)
#define DX(R, p) do { const unsigned l_ = xs + ((p) & 1) * (2 * PIECE) + (R) * PIECE, x_ = xoff(p); \
                      if (staged) { dma2_sys<0>(l_, (R) ? sx1 : sx0, x_, vx); dma2_sys<1>(l_, (R) ? sx1 : sx0, x_, vx); } \
                      else { dma2<0>(l_, (R) ? sx1 : sx0, x_, vx); dma2<1>(l_, (R) ? sx1 : sx0, x_, vx); } } while (0)
#define DW4(s, slot) do { const unsigned o_ = woff(s); DW(0, o_, slot); DW(1, o_, slot); DW(2, o_, slot); DW(3, o_, slot); } while (0)
#define SB __builtin_amdgcn_sched_barrier(0)
    float* outp = dg ? peerout : myout;
    unsigned scv;
    SB;
    asm volatile("s_nop 4");
#define NW4(s) do { const unsigned o_ = woff(s), l_ = wsb + ((s) & 3) * PIECE; dma2<0>(l_, srw, o_, vn); dma2<1>(l_, srw, o_, vn); } while (0)
#define NX4(R, p, slot) do { const unsigned o_ = xoff(p), l_ = wsb + (4 + (R)) * PIECE + (slot); \
                             dma2<0>(l_, (R) ? sx1 : sx0, o_, vx); dma2<1>(l_, (R) ? sx1 : sx0, o_, vx); } while (0)
    if (nar) { NX4(0, 0, 0); if (full) NX4(1, 0, 0); NW4(0); NX4(0, 1, 2 * PIECE); NW4(1); if (full) NX4(1, 1, 2 * PIECE); NW4(2); }
    else { if (!XF) DW4(0, 0); DX(0, 0); if (full) DX(1, 0); if (XF) DW4(0, 0); DW4(1, 1); DX(0, 1); if (EW2) DW4(2, 2); }
    SB;
    const unsigned rw = 8 * wid << nar, ra = row0 + rw, rc = ra + 32;
    unsigned ln = lane;
    asm volatile("" : "+v"(ln));
    const unsigned rl = ln >> (5 - nar), cl = (ln & (31 >> nar)) * 4;
    const unsigned oa = staged ? ((rank + ra / FROWS + 1) & 7) * MS + RROWS + ra % FROWS : rank * MS + ra;
    const unsigned oc = staged ? ((rank + rc / FROWS + 1) & 7) * MS + RROWS + rc % FROWS : rank * MS + rc;
    float* ob = outp + (size_t)rl * NS + co + cl;
    op0 = ob + (size_t)oa * NS;
    float* op1 = ob + (size_t)oc * NS;
    unsigned nr = ((rank + dg) & 7) < nranks ? (rw < nrows) + (!nar && rw + 32 < nrows) : 0;
    unsigned pr = ldsb + (rw + rl) * PROW + cl * 4, pq = pr + 2 * WAVE_LDS;
    nr = rfl(nr);
    asm volatile("" : "+v"(op0), "+v"(op1), "+v"(pr), "+v"(pq), "+s"(nr));
    SB;
    STAMP(10);
    auto read_w = [&](unsigned slot, int n, v8i (&wf)[4]) {
      unsigned ad = wa[n] + slot * (2 * PIECE);
      v2i t0 = __builtin_amdgcn_ds_read_tr8_b64_v2i32((LAS v2i*)(size_t)(ad));
      v2i t1 = __builtin_amdgcn_ds_read_tr8_b64_v2i32((LAS v2i*)(size_t)(ad + 1024));
      v2i t2 = __builtin_amdgcn_ds_read_tr8_b64_v2i32((LAS v2i*)(size_t)(ad + 2048));
      v2i t3 = __builtin_amdgcn_ds_read_tr8_b64_v2i32((LAS v2i*)(size_t)(ad + 3072));
      wf[n] = v8i{t0[0], t0[1], t1[0], t1[1], t2[0], t2[1], t3[0], t3[1]};
    };
    auto read_n = [&](unsigned s, v8i (&wf)[4]) {
#pragma unroll
      for (int n = 0; n < 2; ++n) {
        const unsigned ad = (na ^ (n << 5)) + (s & 3) * PIECE;
        v2i t0 = __builtin_amdgcn_ds_read_tr8_b64_v2i32((LAS v2i*)(size_t)(ad));
        v2i t1 = __builtin_amdgcn_ds_read_tr8_b64_v2i32((LAS v2i*)(size_t)(ad + 512));
        v2i t2 = __builtin_amdgcn_ds_read_tr8_b64_v2i32((LAS v2i*)(size_t)(ad + 1024));
        v2i t3 = __builtin_amdgcn_ds_read_tr8_b64_v2i32((LAS v2i*)(size_t)(ad + 1536));
        wf[n] = v8i{t0[0], t0[1], t1[0], t1[1], t2[0], t2[1], t3[0], t3[1]};
      }
    };
    auto read_x = [&](unsigned odd, unsigned slot, v8i (&xf)[2], int nb) {
      const unsigned ad = (xa ^ (odd << 6)) + slot;
#pragma unroll
      for (int r = 0; r < nb; ++r) {
        v4i lo = *(const LAS v4i*)(size_t)(ad + r * PIECE);
        v4i hi = *(const LAS v4i*)(size_t)((ad ^ 16) + r * PIECE);
        xf[r] = v8i{lo[0], lo[1], lo[2], lo[3], hi[0], hi[1], hi[2], hi[3]};
      }
    };
#define XSL(s) ((((s) >> 1) & 1) * (2 * PIECE))
#define MM(Z, r, n, wf, xf) acc[r][n] = __builtin_amdgcn_mfma_scale_f32_32x32x64_f8f6f4(wf[n], xf[r], (Z) ? v16f{} : acc[r][n], 0, 0, 0, 0, 0, 0)
#define STEP(Z, s, ODD, WAIT, IW, wC, xC, wN, xN) do { \
      const unsigned sn = sl == 2 ? 0 : sl + 1; \
      const unsigned o_ = woff((s) + 3); \
      SB; MM(Z, 0, 0, wC, xC); SB; \
      if (IW) DX(!(ODD), ((s) + 3) >> 1); \
      SB; MM(Z, 0, 1, wC, xC); SB; \
      if (IW) { DW(0, o_, sl); DW(1, o_, sl); } \
      SB; MM(Z, 0, 2, wC, xC); SB; \
      if (IW) { DW(2, o_, sl); DW(3, o_, sl); } \
      SB; MM(Z, 0, 3, wC, xC); SB; \
      vmwait<(WAIT) + ((IW) ? 12 : 0)>(); \
      read_w(sn, 0, wN); read_w(sn, 1, wN); \
      SB; MM(Z, 1, 0, wC, xC); SB; \
      read_w(sn, 2, wN); read_w(sn, 3, wN); \
      SB; MM(Z, 1, 1, wC, xC); SB; \
      read_x(!(ODD), XSL((s) + 1), xN, 2); \
      SB; MM(Z, 1, 2, wC, xC); SB; \
      SB; MM(Z, 1, 3, wC, xC); SB; \
      asm volatile("s_waitcnt lgkmcnt(0)" ::: "memory"); \
      sl = sn; \
      SB; } while (0)
#define HX(H, s) dma2<H>(xs + XSL((s) + 3), sx0, xoff(((s) + 3) >> 1), vx)
#define HSTEP(Z, s, ODD, WAIT, IW, wC, xC, wN, xN) do { \
      const unsigned sn = sl == 2 ? 0 : sl + 1; \
      const unsigned o_ = woff((s) + 3); \
      SB; MM(Z, 0, 0, wC, xC); SB; \
      if (IW && ODD) { HX(0, s); HX(1, s); } \
      if (IW) { DW(0, o_, sl); DW(1, o_, sl); } \
      SB; MM(Z, 0, 1, wC, xC); SB; \
      if (IW) { DW(2, o_, sl); DW(3, o_, sl); } \
      vmwait<(IW) ? 20 : (WAIT)>(); \
      read_w(sn, 0, wN); read_w(sn, 1, wN); \
      SB; MM(Z, 0, 2, wC, xC); SB; \
      read_w(sn, 2, wN); read_w(sn, 3, wN); read_x(!(ODD), XSL((s) + 1), xN, 1); \
      SB; MM(Z, 0, 3, wC, xC); SB; \
      asm volatile("s_waitcnt lgkmcnt(0)" ::: "memory"); \
      sl = sn; \
      SB; } while (0)
#define PSTAMP
#define LAST(NB, NN) do { \
      _Pragma("unroll") for (int t = 0; t <= NB * NN; ++t) { \
        SB; \
        if (t < NB * NN) MM(0, t / NN, t % NN, wB, xB); \
        SB; \
        if (t) _Pragma("unroll") for (int i = 0; i < 4; ++i) { \
          const v16f& c_ = acc[(t - 1) / NN][(t - 1) % NN]; \
          *(LAS v4f*)(size_t)(pw + (t - 1) / NN * 32 * PROW + (t - 1) % NN * 128 + i * 32) = v4f{c_[4 * i], c_[4 * i + 1], c_[4 * i + 2], c_[4 * i + 3]}; \
        } \
      } } while (0)
#define KLOOP(ST, NB) do { \
      v16f acc[NB][4]; \
      v8i wA[4], wB[4], xA[2], xB[2]; \
      asm volatile("s_waitcnt vmcnt(%2)" :: "v"(cpf), "v"(cv), "n"(EW2 ? 20 : 12) : "memory");     \
      EVT(__builtin_amdgcn_s_memrealtime()); \
      read_w(0, 0, wA); read_w(0, 1, wA); read_w(0, 2, wA); read_w(0, 3, wA); read_x(0, 0, xA, NB); \
      SB; if (!EW2) DW4(2, 2); \
      asm volatile("s_waitcnt lgkmcnt(0)" ::: "memory"); \
      unsigned sl = 0;        \
      ST(1, 0, 0, 12, 1, wA, xA, wB, xB); \
      _Pragma("unroll 1") \
      for (unsigned p = 0; p < 6; ++p) { \
        PSTAMP; \
        ST(0, 2 * p + 1, 1, 8, 1, wB, xB, wA, xA); \
        ST(0, 2 * p + 2, 0, 12, 1, wA, xA, wB, xB); \
      } \
      asm volatile("global_load_dword %0, %1, off sc1" : "=v"(scv) : "v"(psc) : "memory"); \
      ST(0, 13, 1, 9, 0, wB, xB, wA, xA); \
      ST(0, 14, 0, 1, 0, wA, xA, wB, xB); \
      LAST(NB, 4); \
    } while (0)
#define NSTEP(NB, Z, s, ODD, WA, WB, IW, IX, wC, xC, wN, xN) do { \
      const unsigned xn = xc == 4 * PIECE ? 0 : xc + 2 * PIECE, xp = xc ? xc - 2 * PIECE : 4 * PIECE; \
      SB; MM(Z, 0, 0, wC, xC); SB; \
      if (IX && (ODD || NB == 2)) NX4(!(ODD), ((s) + 5) >> 1, (ODD) ? xc : xp); \
      if (NB == 2) { SB; MM(Z, 0, 1, wC, xC); SB; } \
      if (IW) NW4((s) + 4); \
      vmwait<NB == 2 ? WA : WB>(); read_n((s) + 1, wN); \
      if (NB == 2) { SB; MM(Z, 1, 0, wC, xC); SB; } \
      read_x(!(ODD), ((ODD) ? xn : xc) - 2 * PIECE, xN, NB); \
      SB; MM(Z, NB - 1, 1, wC, xC); SB; \
      asm volatile("s_waitcnt lgkmcnt(0)" ::: "memory"); \
      if (ODD) xc = xn; \
      SB; } while (0)
#define NLOOP(NB) do { \
      v16f acc[NB][2]; \
      v8i wA[4], wB[4], xA[2], xB[2]; \
      asm volatile("s_waitcnt vmcnt(%1)" :: "v"(cpf), "n"(8 + 4 * NB), "v"(cv) : "memory"); \
      read_n(0, wA); read_x(0, -2 * PIECE, xA, NB); \
      SB; NX4(0, 2, 4 * PIECE); NW4(3); \
      asm volatile("s_waitcnt lgkmcnt(0)" ::: "memory"); \
      unsigned xc = 0; \
      NSTEP(NB, 1, 0, 0, 24, 16, 1, 1, wA, xA, wB, xB); \
      _Pragma("unroll 1") \
      for (unsigned p = 0; p < 5; ++p) { \
        PSTAMP; \
        NSTEP(NB, 0, 2 * p + 1, 1, 24, 20, 1, 1, wB, xB, wA, xA); \
        NSTEP(NB, 0, 2 * p + 2, 0, 24, 16, 1, 1, wA, xA, wB, xB); \
      } \
      NSTEP(NB, 0, 11, 1, 20, 16, 1, 0, wB, xB, wA, xA); \
      asm volatile("global_load_dword %0, %1, off sc1" : "=v"(scv) : "v"(psc) : "memory"); \
      NSTEP(NB, 0, 12, 0, 13, 9, 0, 0, wA, xA, wB, xB); \
      NSTEP(NB, 0, 13, 1, 5, 5, 0, 0, wB, xB, wA, xA); \
      NSTEP(NB, 0, 14, 0, 1, 1, 0, 0, wA, xA, wB, xB); \
      LAST(NB, 2); \
    } while (0)
    if (NAR && nar) { if (full) NLOOP(2); else NLOOP(1); }
    else if (full) KLOOP(STEP, 2); else KLOOP(HSTEP, 1);
    SB;
    STAMP(1);
    unsigned long long tb;
    asm volatile("s_memrealtime %1\n\ts_waitcnt vmcnt(0)" : "+v"(scv), "=s"(tb) :: "memory");
    float scale = bad ? __builtin_nanf("") : __uint_as_float(__builtin_amdgcn_readlane(scv, 0)) * __uint_as_float(__builtin_amdgcn_readlane(scv, 1));
    call = __builtin_amdgcn_readlane(scv, 2) + 1;
    unsigned slot = dg ? slots & 0x7fff : 0, win = dg ? slots & 0x8000 : 0, fl = flag && !dg;
    slot = rfl(slot); win = rfl(win); fl = rfl(fl);
    asm volatile("s_waitcnt lgkmcnt(0)\n\ts_barrier" : "+v"(scale), "+s"(tb), "+s"(slot), "+s"(win), "+s"(fl), "+s"(call) :: "memory");
    if (__builtin_expect(fl, 0)) {
      if (threadIdx.x == 0) __hip_atomic_store(&peerws[WS_FLAG + (8 - grp) * 32 + j], call, __ATOMIC_RELAXED, __HIP_MEMORY_SCOPE_SYSTEM);
      flag = false; LSTAMP(1);
    }
    if (__builtin_expect((unsigned)(tb - t_start) < slot, 0))
      while ((unsigned)(__builtin_amdgcn_s_memrealtime() - t_start) < slot) __builtin_amdgcn_s_sleep(2);
    float* op = op0;
#pragma unroll 1
    for (unsigned r = 0; r < nr; ++r) {
      v4f v[4], p[4][4];
      SB;
#pragma unroll
      for (int i = 0; i < 4; ++i) {
        const unsigned o = i * 2 * PROW << nar;
        p[i][0] = *(const LAS v4f*)(size_t)(pr + o);
        p[i][1] = *(const LAS v4f*)(size_t)(pr + o + WAVE_LDS);
        p[i][2] = *(const LAS v4f*)(size_t)(pq + o);
        p[i][3] = *(const LAS v4f*)(size_t)(pq + o + WAVE_LDS);
        SB;
      }
      pr += 32 * PROW; pq += 32 * PROW;
#pragma unroll
      for (int i = 0; i < 4; ++i) {
        v[i] = (p[i][0] + p[i][1]) + (p[i][2] + p[i][3]);
#pragma unroll
        for (int e = 0; e < 4; ++e) {
          float y = v[i][e] * scale;
          float yc = fminf(fmaxf(y, -60.0f), 60.0f);
          v[i][e] = y * __builtin_amdgcn_rcpf(1.0f + __expf(-yc));
        }
        asm volatile("" : "+v"(v[i]));
        if (WIN < 8 && 4 + i >= WIN && win) {
          vmwait<WIN - 1>();
        }
        asm volatile("global_store_dwordx4 %0, %1, off" PS "\n\ts_nop 1" :: "v"(op + (i * 2 * NS << nar)), "v"(v[i]) : "memory");
        SB;
      }
      op = op1;
    }
    desc >>= 16;
    slots >>= 16; starts >>= 16;
    if (desc) asm volatile("s_barrier" ::: "memory");
  }
  asm volatile("s_waitcnt vmcnt(0)\n\ts_barrier" ::: "memory");
  STAMP(4);
  if (wid == 0) {
    if (lane == 0) {
      if (flag) __hip_atomic_store(&peerws[WS_FLAG + (8 - grp) * 32 + j], call, __ATOMIC_RELAXED, __HIP_MEMORY_SCOPE_SYSTEM);
      __hip_atomic_store(&myws[WS_CNT + bid], call, __ATOMIC_RELAXED, __HIP_MEMORY_SCOPE_AGENT);
    }
    if (flag) LSTAMP(1);
    LSTAMP(2);
    if (bid == WAITER) {
      const unsigned so = 1 + (lane >> 3);
      const unsigned* f = myws + WS_FLAG + so * 32 + (lane & 7) * 4;
      unsigned long long t0 = __builtin_amdgcn_s_memrealtime();
      bool pend = lane < 56 && ((rank + so) & 7) < nranks;
      bool timed_out = false;
      while (true) {
        if (pend) {
          v4u fv;
          asm volatile("global_load_dwordx4 %0, %1, off sc0 sc1\n\ts_waitcnt vmcnt(0)" : "=v"(fv) : "v"(f) : "memory");
          pend = (int)(fv[0] - call) < 0 || (int)(fv[1] - call) < 0 || (int)(fv[2] - call) < 0 || (int)(fv[3] - call) < 0;
        }
        if (__builtin_amdgcn_ballot_w64(pend) == 0) break;
        if (__builtin_amdgcn_s_memrealtime() - t0 > DEADLINE_TICKS) { timed_out = true; break; }
      }
      if (timed_out && lane == 0) {
        __hip_atomic_store(&myws[WS_TMO], call, __ATOMIC_RELAXED, __HIP_MEMORY_SCOPE_AGENT);
        *op0 = __builtin_nanf("");
      }
      __builtin_amdgcn_fence(__ATOMIC_ACQUIRE, "");
      STAMP(15); LSTAMP(3);
    }
  }
}

#define PI(n) (n)
#define PT(n) (n)
#define CX constexpr
struct Tile { int cb, row, rows, half; };
static CX int tiles(Tile* t) {
  int end[8] = {RROWS, RROWS, RROWS, RROWS, RROWS, RROWS, RROWS, RROWS}, beg[8] = {}, n = 0;
  for (const char* c = PT(TL); *c; ++c, ++n) {
    const int l = (*c | 32) - 'a', cb = l & 3, hf = l > 3 && beg[cb + 4] < beg[cb] ? cb + 4 : cb, half = l > 3 ? 1 + hf / 4 : 0;
    const int row = (end[hf] - 1) / 32 * 32 - (*c & 32 ? 0 : 32), top = beg[hf] + (*c & 32 ? 32 : 64);
    t[n] = PI(UP) ? Tile{cb, beg[hf], (top < RROWS ? top : RROWS) - beg[hf], half} : Tile{cb, row, end[hf] - row, half};
    end[hf] = row; beg[hf] = top;
    if (!half) { end[cb + 4] = row; beg[cb + 4] = top; }
  }
  return n;
}
static CX void build_desc(unsigned* desc, unsigned (*slot)[2] = nullptr) {
  auto T32 = [](unsigned staged, unsigned rb, int rows, unsigned cb, unsigned remote, unsigned rot) {
    return staged | rb << 1 | remote << 7 | cb << 8 | (rows / 8 - 1) << 10 | (rot & 7) << 13;
  };
  auto T = [&](unsigned staged, unsigned rp, int rows, unsigned cb, unsigned remote, unsigned rot) {
    return T32(staged, 2 * rp, rows < 64 ? rows : 64, cb, remote, rot);
  };
  Tile tl[64] = {};
  const int nt = tiles(tl), NX = nt - NRW;
  constexpr int NFP = (7 * FROWS + 63) / 64;
  int used[8] = {};
  auto second = [&](int g, unsigned t) {
    int i = used[g]++;
    if (i < NFW) desc[i * 8 + g] = t; else desc[(i + NX) * 8 + g] |= t << 16;
  };
  if (slot) for (int i = 0; i < 32; ++i) { slot[i][0] = 0; slot[i][1] = PI(LST) * (i < NFW ? 1 : 0x10000); }
  for (int n = 0, at = 0; n < nt; ++n) {
    const int w = NFW + n % NRW, sh = n < NRW ? 0 : 16;
    for (int g = 1; g < 8; ++g) {
      const unsigned t = T32(tl[n].half > 0, tl[n].row / 32, tl[n].rows, tl[n].cb, 1, XROT * g) | (tl[n].half & 2) << 5;
      if (sh) desc[w * 8 + g] |= t << 16; else desc[w * 8 + g] = t;
    }
    if (slot) {
      const int hold = n >= PI(SLN) ? (n - PI(SLN) + 1) * PI(SLS) : 0;
      const int t = PI(SLOT0) + at * PI(SLOTK) / 128 + (hold < PI(SLD) ? hold : PI(SLD));
      int st = (n >= PI(LN) ? t - (tl[n].rows > 32 ? PI(LEAD) : PI(LEADH)) - (n - PI(LN)) * PI(LEADS) : 0) + (PT(AT)[n] - 'm') * 20;
      if (st < 0) st = 0;
      slot[w][0] |= (unsigned)(t + (PT(AS)[n] - 'm') * 8 + (n >= PI(WN0) && n < PI(WN1) ? 0x8000 : 0)) << sh;
      slot[w][1] = (slot[w][1] & (0xffff0000u >> sh)) | st << sh;
    }
    at += tl[n].half ? tl[n].rows / 2 : tl[n].rows;
  }
  bool out[4][16] = {};
  for (int pass = 0; pass < 2; ++pass) {
    if (pass == PI(FSW)) for (int n = 31; n >= 0; --n) { const int cb = n & 3, rp = n >> 2; second(cb < 3 && rp >= 4 ? cb + 5 : cb + 1, T(0, rp, 64, cb, 0, 2 * rp + rp / 4)); }
    else for (int i = 0, rp = 0; i < 4 * NFP - 32; ++i, ++rp) {
      const int g = i % 6 + 1 + (i % 6 >= 3), cb = (g - 1) & 3;
      while (out[cb][rp % NFP]) ++rp;
      const int r = rp % NFP;
      out[cb][r] = true;
      second(g, T(1, r, 7 * FROWS - 64 * r, cb, 0, r + 2));
    }
  }
  int n = 31;
  for (int rp = NFP - 1; rp >= 0; --rp)
    for (int cb = 0, k = 0; cb < 4; ++cb)
      if (!out[cb][rp]) { desc[n * 8] = T(1, rp, 7 * FROWS - 64 * rp, cb, 0, cb == 3 ? rp + ZR : rp + 4 * k++); --n; }
}
static CX int build_pf(unsigned short (*pf)[16]) {
  Tile tl[64] = {};
  const int nt = tiles(tl), na = PI(NA);
  int due[36][8] = {};
  for (int i = 0; i < 36 * 8; ++i) due[i >> 3][i & 7] = 1 << 30;
  for (int n = 0; n < nt; ++n) {
    const int e = n < na ? 0 : n < PI(LN) ? PI(SH1) + (n - na) * PI(SHS) : PI(SH2) + (n - PI(LN)) * PI(SHT);
    for (int q = n < na ? PI(Q0) : n < PI(LN) ? PI(Q1) : 0; q < 8; ++q) {
      const int t = (e + q * (tl[n].half ? (tl[n].rows > 32 ? PI(DQQ) : PI(DQS)) : tl[n].rows > 32 ? PI(DQ64) : 10)) * 64;
      for (int r = tl[n].row / 16; r * 16 < tl[n].row + tl[n].rows; ++r) { const int u = n < PI(QX) && !q ? -1 : t + (PI(XO) ? 31 - r : r); if (u < due[r][q]) due[r][q] = u; }
      const int u = t + PI(WF) + tl[n].cb;
      if (u < due[32 + tl[n].cb][q] && !(PI(PF3) && tl[n].cb == 3)) due[32 + tl[n].cb][q] = u;
    }
  }
  for (int r = 0; r < 32; ++r) if (due[r][0] < 0) due[r][0] = 1 << 30;
  unsigned short job[1024] = {};
  int nj = 0;
  while (true) {
    int best = 0;
    for (int i = 1; i < 36 * 8; ++i) if (due[i >> 3][i & 7] < due[best >> 3][best & 7]) best = i;
    const int r = best >> 3, q = best & 7;
    if (due[r][q] == 1 << 30) break;
    due[r][q] = 1 << 30;
    if (r < 32) job[nj++] = 1 | q << 3 | r << 6;
    else for (int i = 0; i < 8; ++i) job[nj++] = (r - 30) | q << 3 | i << 6;
  }
  int k = 0;
  const char* rd = PT(RD);
  for (int i = 0; i < MPF * NW * 16; ++i) pf[i >> 4][i & 15] = 0;
  for (int n = 0; n < 16; ++n)
    for (int i0 = 0; i0 < (NRW - na) * NW; ++i0) {
      const int i = n < PI(DO) ? i0 % (NRW - na) * NW + i0 / (NRW - na) : i0;
      const int c = rd[i / NW], rounds = c > '9' ? c - 'a' + 10 : c - '0';
      pf[(NRW - na - 1 - i / NW) * NW + i % NW][n] = n < rounds && k < nj ? job[k++] : 0;
    }
  return nj - k;
}
static CX Args tables() {
  Args a = {};
  a.cfg = (NFW + PI(NA)) | PI(BADR) << 8 | PI(PF3) << 10 | PI(DLY0) << 16;
  a.cfg2 = PI(PT0) | PI(PTW) << 9 | PI(PTR) << 16 | (PI(PT0) ? (unsigned)PI(PTN) : 0u) << 26;
  build_desc(a.desc, a.slot);
  build_pf(a.pf);
  return a;
}

extern "C" void kernel_launch(int rank, int nranks, ncclComm_t comm, hipStream_t stream, void* const* d_in, const int* in_sizes, int n_in,
                              void* d_out, int out_size, const int* in_dtypes, int out_dtype, void* const* d_out_all,
                              void* const* d_ws_all, size_t ws_bytes) {
  if (nranks != NR || rank < 0 || rank >= NR || n_in < 4 || in_sizes[0] != MS * KD || in_sizes[1] != KD * ND || out_size != NR * MS * NS ||
      ws_bytes < WS_STAGE + 3 * (size_t)STAGE_BYTES || ((size_t)d_ws_all[rank] & 4095)) {
    fprintf(stderr, "unexpected problem shape\n");
    return;
  }
  CX Args init = tables();
  Args a = init;
  a.x = (const unsigned char*)d_in[0]; a.w = (const unsigned char*)d_in[1];
  a.sx = (const float*)d_in[2]; a.sw = (const float*)d_in[3];
  for (int i = 0; i < NR; ++i) { a.peer[i].out = (float*)d_out_all[(rank + i) % NR]; a.peer[i].ws = (unsigned*)d_ws_all[(rank + i) % NR]; }
  a.rank = rank; a.nranks = nranks;
  gemm_a2a<<<256, 256, 0, stream>>>(a);
}
```
